```python
import numpy as np

kernel_name = "gpu_gemm_ar_m4096_k4096_n16384_f32_reluqe4m3"


ABSMAX_RTOL = 0.09

COMPUTE_DTYPE = "float32"
EPILOGUE = "relu_quant_e4m3"
IN_QUANT = None
OUT_QUANT = "e4m3"
IN_DTYPES = {"x": "float32", "w_mat": "float32"}
WORLD_SIZE = 8

def setup_inputs():

    x = np.random.randn(4096, 4096).astype(np.float32)
    w_mat = (np.random.randn(4096, 16384) / np.sqrt(4096)).astype(np.float32)
    return {"x": x, "w_mat": w_mat}

def _snap_fp8(x, mant_bits, max_abs, min_normal):
    a = np.abs(x.astype(np.float64))
    _m, _e = np.frexp(a)
    q = np.where(a >= min_normal, np.ldexp(1.0, _e - 1 - mant_bits),
                 min_normal * 2.0 ** -mant_bits)
    snapped = np.minimum(np.round(a / q) * q, max_abs)
    return (np.sign(x) * snapped).astype(np.float32)
def _quant_dequant(y, amax):
    scale = amax / 448.0
    q = _snap_fp8(y / scale, 3, 448.0, 2.0 ** -6)
    return (q * scale).astype(np.float32)
def _epilogue(y):
    y = np.maximum(y, 0.0)
    return _quant_dequant(y, np.abs(y).max())

def reference(x, w_mat):
    y = (x @ w_mat).astype(np.float32)
    return _epilogue(y)
```

```cpp
#include <hip/hip_runtime.h>
#include <rccl/rccl.h>
#include <cstdio>
#include <cstdint>

#define HIP_CHECK(expr) do { hipError_t _e = (expr); if (_e != hipSuccess) { \
    fprintf(stderr, "HIP error %s at %s:%d\n", hipGetErrorString(_e), __FILE__, __LINE__); \
    return; } } while (0)

namespace {

#define LAS __attribute__((address_space(3)))
#define GAS __attribute__((address_space(1)))
typedef short bf16x8 __attribute__((ext_vector_type(8)));
typedef float f32x4 __attribute__((ext_vector_type(4)));
typedef unsigned u32x4 __attribute__((ext_vector_type(4)));
typedef unsigned u32x2 __attribute__((ext_vector_type(2)));
typedef __bf16 bf16x2_t __attribute__((ext_vector_type(2)));
typedef GAS unsigned gu32;
typedef GAS unsigned long long gu64;

constexpr int WORLD = 8;
constexpr int MM = 4096, KK = 4096, NN = 16384;
constexpr int KS = KK / WORLD;
constexpr int NS = NN / WORLD;
constexpr int LEVELS = KS / 64;
constexpr int NTILES = KK / 64;
constexpr int NWG = 256;

constexpr size_t KiB = 1024, MiB = 1024 * 1024;
constexpr size_t OFF_TICKET = 0;
constexpr size_t OFF_TMO = 256;
constexpr size_t OFF_CNT = 1 * KiB;
constexpr size_t OFF_CNT0 = 2 * KiB + 512;
constexpr size_t OFF_QCNT = 2 * KiB;
constexpr size_t OFF_RDY = 3 * KiB;
constexpr size_t OFF_INFLAG = 4 * KiB;
constexpr size_t OFF_AMAX = 40 * KiB;
constexpr size_t OFF_PKFLAG = 48 * KiB;
constexpr size_t OFF_XS = 1 * MiB;
constexpr size_t XS_BYTES = 32 * MiB;
constexpr size_t OFF_WS = OFF_XS + XS_BYTES;
constexpr size_t WS_BYTES = 16 * MiB;
constexpr size_t OFF_PK = OFF_WS + WS_BYTES;
constexpr size_t PK_TILE = 72 * KiB;
constexpr size_t OFF_SBX = OFF_PK + 128 * PK_TILE;
constexpr size_t SBX_BYTES = 4 * MiB;
constexpr size_t OFF_SBW = OFF_SBX + SBX_BYTES;
constexpr size_t SBW_BYTES = 16 * MiB;
constexpr size_t OFF_MK = OFF_SBW + SBW_BYTES;
constexpr size_t WS_NEEDED = OFF_MK + 8 * MiB;
constexpr size_t OUT_BYTES = (size_t)MM * NN;

constexpr unsigned long long DEADLINE_TICKS = 150000000ull;

constexpr int HTB = 16384;
constexpr int LDS_STAGE = 8 * HTB;
constexpr int LDS_CTRL = LDS_STAGE;
constexpr int LDS_FLAG = LDS_CTRL;
constexpr int LDS_MISC = LDS_CTRL + 512;
constexpr int LDS_CALL = LDS_CTRL + 2044;
constexpr int LDS_LUTS = LDS_CTRL + 2048;
constexpr int LDS_LUTR = LDS_CTRL + 2560;
constexpr int LDS_BYTES = LDS_CTRL + 3072;
constexpr int EX_MASK = 0;
constexpr int EX_RING = 56 * 1024;

constexpr int AUX_SYS = 17;
#define MASK_PUSH 1
#define HL0_W 1
#define PULL_W1 2
#define PULL_W 4

struct Params {
    const float* x;
    const float* w;
    unsigned char* out[WORLD];
    unsigned char* ws[WORLD];
    int rank;
};

__device__ __forceinline__ __amdgpu_buffer_rsrc_t mkrsrc(const void* p, unsigned bytes) {
    return __builtin_amdgcn_make_buffer_rsrc((void*)p, 0, bytes, 0x00020000);
}
__device__ __forceinline__ unsigned pk_bf16(float a, float b) {
    bf16x2_t v = {(__bf16)a, (__bf16)b};
    return __builtin_bit_cast(unsigned, v);
}
__device__ __forceinline__ unsigned long long rtime() { return __builtin_amdgcn_s_memrealtime(); }
__device__ __forceinline__ unsigned ld_sys(const void* p) { return __hip_atomic_load((const gu32*)p, __ATOMIC_RELAXED, __HIP_MEMORY_SCOPE_SYSTEM); }
__device__ __forceinline__ void st_sys(void* p, unsigned v) { __hip_atomic_store((gu32*)p, v, __ATOMIC_RELAXED, __HIP_MEMORY_SCOPE_SYSTEM); }
__device__ __forceinline__ unsigned ld_agent(const void* p) { return __hip_atomic_load((const gu32*)p, __ATOMIC_RELAXED, __HIP_MEMORY_SCOPE_AGENT); }
__device__ __forceinline__ void st_agent(void* p, unsigned v) { __hip_atomic_store((gu32*)p, v, __ATOMIC_RELAXED, __HIP_MEMORY_SCOPE_AGENT); }

__device__ __forceinline__ bool mask_pushed(int s, int q, int t) { return (((q - s - 1) & 7) + 7 - t % 7) % 7 < MASK_PUSH; }
__device__ __forceinline__ unsigned pack_rows(unsigned cnt) { return ((cnt < 8192u ? cnt : 8192u) + 127u) >> 7; }

__device__ __forceinline__ unsigned wave_scan(unsigned x) {
    x += (unsigned)__builtin_amdgcn_update_dpp(0, (int)x, 0x111, 0xf, 0xf, false);
    x += (unsigned)__builtin_amdgcn_update_dpp(0, (int)x, 0x112, 0xf, 0xf, false);
    x += (unsigned)__builtin_amdgcn_update_dpp(0, (int)x, 0x114, 0xf, 0xf, false);
    x += (unsigned)__builtin_amdgcn_update_dpp(0, (int)x, 0x118, 0xf, 0xf, false);
    x += (unsigned)__builtin_amdgcn_update_dpp(0, (int)x, 0x142, 0xa, 0xf, false);
    x += (unsigned)__builtin_amdgcn_update_dpp(0, (int)x, 0x143, 0xc, 0xf, false);
    return x;
}
__device__ __forceinline__ void init_luts(LAS unsigned char* lds, int t) {
    const int r = t >> 4, m4 = t & 15;
    unsigned lo = 0x0c0c0c0cu, hi = 0x0c0c0c0cu, ex = 0x0c0c0c0cu;
    int pos = r;
#pragma unroll
    for (int f = 0; f < 4; ++f) {
        if ((m4 >> (3 - f)) & 1) {
            if (pos < 4) lo = (lo & ~(0xffu << (8 * pos))) | ((unsigned)f << (8 * pos));
            else hi = (hi & ~(0xffu << (8 * (pos - 4)))) | ((unsigned)f << (8 * (pos - 4)));
            ex = (ex & ~(0xffu << (8 * f))) | ((unsigned)pos << (8 * f));
            ++pos;
        }
    }
    *(LAS u32x2*)(lds + LDS_LUTS + t * 8) = u32x2{lo, hi};
    *(LAS unsigned*)(lds + LDS_LUTR + t * 4) = ex;
}

__device__ __forceinline__ bool wait_giveup(unsigned char* myws, unsigned call, unsigned long long t0, unsigned& it) {
    if ((++it & 7u) == 0u) {
        const unsigned tm = __builtin_amdgcn_readfirstlane(ld_agent(myws + OFF_TMO));
        if (tm == call) return true;
        if (rtime() - t0 > DEADLINE_TICKS) { st_agent(myws + OFF_TMO, call); return true; }
    }
    __builtin_amdgcn_s_sleep(1);
    return false;
}
__device__ __forceinline__ bool lds_spin_expired(unsigned& n, unsigned long long& t0) {
    __builtin_amdgcn_s_sleep(1);
    if ((++n & 4095u) != 0u) return false;
    const unsigned long long t = rtime();
    if (t0 == 0ull) { t0 = t; return false; }
    return t - t0 > 2ull * DEADLINE_TICKS;
}
__device__ __forceinline__ void lds_wait_ge(volatile LAS unsigned* w, unsigned v) {
    unsigned n = 0;
    unsigned long long t0 = 0;
    while (*w < v) { if (lds_spin_expired(n, t0)) break; }
}

#define WAIT_V(n) asm volatile("s_waitcnt vmcnt(" #n ")" ::: "memory")
#define WAIT_L(n) asm volatile("s_waitcnt lgkmcnt(" #n ")" ::: "memory")
#define BAR __builtin_amdgcn_s_barrier()
#define SCHED __builtin_amdgcn_sched_barrier(0)
template <int N> __device__ __forceinline__ void wait_vm() { asm volatile("s_waitcnt vmcnt(%0)" :: "i"(N) : "memory"); }

constexpr int HL0 = 1;
constexpr int NUNITS = (16 - HL0) * 48;
__device__ __forceinline__ void unit_decode(unsigned n, int m, int& kind, int& hl, int& j) {
    hl = HL0 + (int)(n / 48u);
    const int r = (int)(n % 48u);
    if (r < 16) { kind = 2; j = r; } else { kind = (r - 16) & 1; j = (((r - 16) >> 1) + 4 * m) & 15; }
}
__device__ __forceinline__ void comm_inputs(const Params& p, LAS unsigned char* lds, int dd, int j, int e, unsigned ticket) {
    const int tid = threadIdx.x, lane = tid & 63;
    const int v = __builtin_amdgcn_readfirstlane(tid >> 6);
    const int rank = p.rank;
    unsigned char* myws = p.ws[rank];
    volatile LAS unsigned* callw = (volatile LAS unsigned*)(lds + LDS_CALL);
    volatile LAS unsigned* pland = (volatile LAS unsigned*)(lds + LDS_MISC + 0);
    volatile LAS unsigned* pseq = (volatile LAS unsigned*)(lds + LDS_MISC + 4);
    volatile LAS unsigned* sdone = (volatile LAS unsigned*)(lds + LDS_MISC + 8);
    volatile LAS unsigned* pend = (volatile LAS unsigned*)(lds + LDS_MISC + 12);
    volatile LAS unsigned* desc = (volatile LAS unsigned*)(lds + LDS_MISC + 16);
    LAS unsigned* hcnt = (LAS unsigned*)(lds + LDS_MISC + 832);
    if (v >= 4) {
        const int c = v & 3, kf = c & 1, rg = c >> 1;
        const int q = (rank + dd) & 7;
        const float* xs = p.x + (size_t)(256 * j + 32 * dd + 16 * rg + (lane & 15)) * KS + kf * 32 + 8 * (lane >> 4);
        const float* wsrc = p.w + (size_t)(kf * 32 + 8 * (lane >> 4)) * NN + q * NS + j * 128 + rg * 64 + 4 * (lane & 15);
        f32x4 lx[2], lw[8];
        auto load_level = [&](int u) {
#pragma unroll
            for (int i = 0; i < 8; ++i) lw[i] = __builtin_nontemporal_load((const f32x4*)(wsrc + (size_t)(u * 64 + i) * NN));
#pragma unroll
            for (int i = 0; i < 2; ++i) lx[i] = __builtin_nontemporal_load((const f32x4*)(xs + u * 64 + 4 * i));
        };
        const int u0 = kf == 0 ? 1 : 0;
        load_level(u0);
        BAR;
        const __amdgpu_buffer_rsrc_t sbx = mkrsrc(myws + OFF_SBX, (unsigned)SBX_BYTES), xsr = mkrsrc(myws + OFF_XS, (unsigned)XS_BYTES);
        const __amdgpu_buffer_rsrc_t wdr = dd != 0 ? mkrsrc(myws + OFF_SBW, (unsigned)SBW_BYTES) : mkrsrc(myws + OFF_WS, (unsigned)WS_BYTES);
        const unsigned kfs = rank & 1, tm = rank >> 1;
        const unsigned sbx0 = (unsigned)((kf * 16 + j) * 16 + 2 * dd + rg) * 1024u + lane * 16u;
        const unsigned xs0 = (unsigned)(4 * kf + tm) * (512u * 1024u) + (unsigned)(16 * j + 2 * dd + rg) * 2048u + kfs * 1024u + lane * 16u;
        const unsigned w0 = dd != 0 ? (unsigned)(((q * 16 + kf) * 16 + j) * 8 + rg * 4) * 1024u + lane * 16u
                                    : (unsigned)(4 * kf + tm) * (256u * 1024u) + (unsigned)(j >> 1) * 32768u + (unsigned)((j & 1) * 2 + rg) * 4096u + kfs * 1024u + lane * 16u;
        const unsigned wstep = dd != 0 ? 2u * 16u * 8192u : 8u * 256u * 1024u;
#pragma unroll 1
        for (int u = u0; u < LEVELS; ++u) {
            const int hl = 2 * u + kf;
            const u32x4 cx = {pk_bf16(lx[0][0], lx[0][1]), pk_bf16(lx[0][2], lx[0][3]), pk_bf16(lx[1][0], lx[1][1]), pk_bf16(lx[1][2], lx[1][3])};
            u32x4 cw[4];
#pragma unroll
            for (int f = 0; f < 4; ++f)
                cw[f] = u32x4{pk_bf16(lw[0][f], lw[1][f]), pk_bf16(lw[2][f], lw[3][f]), pk_bf16(lw[4][f], lw[5][f]), pk_bf16(lw[6][f], lw[7][f])};
#pragma unroll
            for (int f = 0; f < 4; ++f) {
                const unsigned fo = dd != 0 ? (unsigned)f * 1024u : (unsigned)((f >> 1) * 16384 + (f & 1) * 2048);
                __builtin_amdgcn_raw_buffer_store_b128(cw[f], wdr, w0 + (unsigned)u * wstep + fo, 0, AUX_SYS);
            }
            __builtin_amdgcn_raw_buffer_store_b128(cx, sbx, sbx0 + (unsigned)u * (2u * 16u * 16384u), 0, AUX_SYS);
            __builtin_amdgcn_raw_buffer_store_b128(cx, xsr, xs0 + (unsigned)u * (8u * 512u * 1024u), 0, AUX_SYS);
            asm volatile("" ::: "memory");
            if (u + 1 < LEVELS) { load_level(u + 1); asm volatile("" ::: "memory"); wait_vm<10>(); } else wait_vm<0>();
            unsigned old = 0;
            if (lane == 0) old = __hip_atomic_fetch_add(hcnt + hl, 1u, __ATOMIC_RELAXED, __HIP_MEMORY_SCOPE_WORKGROUP);
            if (__builtin_amdgcn_readfirstlane(old) == 1u) {
                unsigned g = 0;
                if (lane == 0) g = __hip_atomic_fetch_add((gu32*)(myws + OFF_CNT + hl * 64), 1u, __ATOMIC_RELAXED, __HIP_MEMORY_SCOPE_AGENT);
                if ((__builtin_amdgcn_readfirstlane(g) & 127u) == 127u) {
                    lds_wait_ge(callw, 1u);
                    const unsigned call = *callw;
                    if (lane < WORLD && lane != rank) st_sys(p.ws[lane] + OFF_RDY + rank * 64 + hl * 4, call);
                    if (lane < 48) st_agent(myws + OFF_INFLAG + (size_t)(rank * 16 + hl) * 256 + lane * 4, call);
                }
            }
        }
        return;
    }
    BAR;
    if (v > 2) return;
    const int link = e < 126 ? e % 7 : e - 126;
    const int s = (rank + 1 + link) & 7;
    volatile LAS unsigned* dq = (volatile LAS unsigned*)(lds + LDS_MISC + 48);
    if (v == 2) {
        lds_wait_ge(callw, 1u);
        const unsigned call = *callw;
        const unsigned per_call = (unsigned)NUNITS + 18u + (link < 2 ? 1u : 0u);
        const unsigned qbase = (call - 1u) * per_call;
        gu32* qp = (gu32*)(myws + OFF_QCNT + link * 64);
        for (unsigned k = 0;; ++k) {
            if (k >= 1) lds_wait_ge(pseq, k);
            unsigned n = 0;
            if (lane == 0) n = __hip_atomic_fetch_add(qp, 1u, __ATOMIC_RELAXED, __HIP_MEMORY_SCOPE_AGENT) - qbase;
            n = __builtin_amdgcn_readfirstlane(n);
            if (n > (unsigned)NUNITS) n = (unsigned)NUNITS;
            if (lane == 0) dq[k & 1u] = n + 1u;
            if (n >= (unsigned)NUNITS) break;
        }
        return;
    }
    if (v == 0) {
        const unsigned call = __builtin_amdgcn_readfirstlane(ticket) / NWG + 1u;
        if (lane == 0) *callw = call;
        const unsigned char* rdy = myws + OFF_RDY + s * 64 + (lane & 15) * 4;
        const unsigned char* sws = p.ws[s];
        bool giveup = false;
        unsigned rmask = 0, issued = 0, vpub = 0, seq = 0;
#pragma unroll 1
        for (;;) {
            lds_wait_ge(dq + (seq & 1u), 1u);
            const unsigned n = __builtin_amdgcn_readfirstlane(dq[seq & 1u]) - 1u;
            if (n >= (unsigned)NUNITS) break;
            int kind, hl, jj;
            unit_decode(n, s >> 1, kind, hl, jj);
            if (!((rmask >> hl) & 1u)) {
                const unsigned long long t0 = rtime();
                unsigned it = 0;
                for (;;) {
                    rmask = (unsigned)__ballot((int)(ld_sys(rdy) - call) >= 0);
                    vpub = issued;
                    if (lane == 0) *pland = vpub;
                    if (((rmask >> hl) & 1u) || giveup) break;
                    giveup = wait_giveup(myws, call, t0, it);
                }
            }
            if (seq >= 8) lds_wait_ge(sdone, seq - 7u);
            if (lane == 0) { desc[seq & 7u] = n; dq[seq & 1u] = 0u; *pseq = seq + 1u; }
            const unsigned char* gb = giveup ? myws : sws;
            const unsigned char* g = (kind == 2 ? gb + OFF_SBW + (size_t)((rank * 16 + hl) * 16 + jj) * 8192
                                                : gb + OFF_SBX + (size_t)((hl * 16 + jj) * 16 + kind * 8) * 1024) + lane * 16;
#pragma unroll
            for (int i = 0; i < 8; ++i) {
                __builtin_amdgcn_global_load_lds((const unsigned*)(g + i * 1024), (LAS unsigned*)(lds + (seq & 7u) * 8192 + i * 1024), 16, 0, AUX_SYS);
                ++issued;
                wait_vm<PULL_W1>();
                if (issued >= (unsigned)PULL_W1 && issued - (unsigned)PULL_W1 > vpub) { vpub = issued - (unsigned)PULL_W1; if (lane == 0) *pland = vpub; }
            }
            ++seq;
        }
        wait_vm<0>();
        if (lane == 0) { *pland = issued; *pend = seq + 1u; }
        return;
    }
    {
        lds_wait_ge(callw, 1u);
        const unsigned call = *callw;
        const __amdgpu_buffer_rsrc_t xsr = mkrsrc(myws + OFF_XS, (unsigned)XS_BYTES), wsr = mkrsrc(myws + OFF_WS, (unsigned)WS_BYTES);
        const unsigned kfs = s & 1, tm = s >> 1;
        unsigned char* flb = myws + OFF_INFLAG + (size_t)s * 4096;
        unsigned char* pending = nullptr;
        for (unsigned seq = 0;; ++seq) {
            unsigned n = 0;
            unsigned long long t0 = 0;
            while (*pland < 8u * (seq + 1u)) {
                if (pending) { wait_vm<0>(); if (lane == 0) st_agent(pending, call); pending = nullptr; }
                const unsigned pe = *pend;
                if (pe != 0u && seq + 1u >= pe) break;
                if (lds_spin_expired(n, t0)) break;
            }
            { const unsigned pe = *pend; if (pe != 0u && seq + 1u >= pe) break; }
            const unsigned un = __builtin_amdgcn_readfirstlane(desc[seq & 7u]);
            int kind, hl, jj;
            unit_decode(un < (unsigned)NUNITS ? un : 0u, s >> 1, kind, hl, jj);
            u32x4 d[8];
#pragma unroll
            for (int i = 0; i < 8; ++i) d[i] = *(const LAS u32x4*)(lds + (seq & 7u) * 8192 + i * 1024 + lane * 16);
            const unsigned T = (unsigned)(4 * hl) + tm;
            if (kind == 2) {
                const unsigned o = T * (256u * 1024u) + (unsigned)(jj >> 1) * 32768u + (unsigned)((jj & 1) * 2) * 4096u + kfs * 1024u + lane * 16u;
#pragma unroll
                for (int i = 0; i < 8; ++i)
                    __builtin_amdgcn_raw_buffer_store_b128(d[i], wsr, o + (unsigned)(((i >> 1) & 1) * 16384 + (i >> 2) * 4096 + (i & 1) * 2048), 0, AUX_SYS);
            } else {
                const unsigned o = T * (512u * 1024u) + (unsigned)(16 * jj + kind * 8) * 2048u + kfs * 1024u + lane * 16u;
#pragma unroll
                for (int i = 0; i < 8; ++i) __builtin_amdgcn_raw_buffer_store_b128(d[i], xsr, o + (unsigned)i * 2048u, 0, AUX_SYS);
            }
            if (lane == 0) *sdone = seq + 1u;
            if (pending) { wait_vm<8>(); if (lane == 0) st_agent(pending, call); }
            pending = flb + hl * 256 + (kind * 16 + jj) * 4;
        }
        if (pending) { wait_vm<0>(); if (lane == 0) st_agent(pending, call); }
    }
}

__device__ __forceinline__ void comm_expand(const Params& p, LAS unsigned char* lds, int e, unsigned call) {
    const int tid = threadIdx.x, lane = tid & 63;
    const int v = __builtin_amdgcn_readfirstlane(tid >> 6);
    const int rank = p.rank;
    unsigned char* myws = p.ws[rank];
    volatile LAS unsigned* mland = (volatile LAS unsigned*)(lds + LDS_MISC + 256);
    volatile LAS unsigned* vland = (volatile LAS unsigned*)(lds + LDS_MISC + 260);
    volatile LAS unsigned* ordr = (volatile LAS unsigned*)(lds + LDS_MISC + 272);
    volatile LAS unsigned* done = (volatile LAS unsigned*)(lds + LDS_MISC + 320);
    volatile LAS unsigned* lens = (volatile LAS unsigned*)(lds + LDS_MISC + 576);
    volatile LAS unsigned* endq = (volatile LAS unsigned*)(lds + LDS_MISC + 1088);
    auto item_src = [&](int k) { return (rank + 1 + (e < 126 ? e % 7 : k)) & 7; };
    auto item_tile = [&](int k) { return e < 126 ? e / 7 + 18 * k : e; };
    if (v == 0) {
        const int fk = (lane & 7) < 7 ? (lane & 7) : 0;
        const unsigned char* fl = myws + OFF_PKFLAG + (size_t)item_tile(fk) * 128 + item_src(fk) * 4 + ((lane >> 3) & 1) * 32;
        bool giveup = false;
        unsigned known = 0;
        auto poll = [&]() { known |= (unsigned)__ballot((int)(ld_sys(fl) - call) >= 0); };
        volatile LAS unsigned* fsamp = (volatile LAS unsigned*)(lds + LDS_FLAG);
        auto sample = [&]() { __builtin_amdgcn_global_load_lds((const unsigned*)fl, (LAS unsigned*)(lds + LDS_FLAG), 4, 0, AUX_SYS); };
        auto absorb = [&]() { known |= (unsigned)__ballot((int)(fsamp[lane] - call) >= 0); };
        unsigned taken = 0, order = 0;
#pragma unroll 1
        for (int k = 0; k < 7; ++k) {
            unsigned avail = known & 0x7fu & ~taken;
            if (avail == 0u) {
                const unsigned long long t0 = rtime();
                unsigned it = 0;
                for (;;) {
                    poll();
                    avail = known & 0x7fu & ~taken;
                    if (avail) break;
                    if (giveup) { avail = 0x7fu & ~taken; break; }
                    giveup = wait_giveup(myws, call, t0, it);
                }
                if (lane == 0) *mland = (unsigned)k;
            }
            const int item = __builtin_ctz(avail);
            taken |= 1u << item;
            order |= (unsigned)item << (4 * k);
            if (lane == 0) ordr[k] = (unsigned)item;
            const unsigned char* g = (mask_pushed(item_src(item), rank, item_tile(item))
                                          ? myws + OFF_MK + (size_t)(item_src(item) * 128 + item_tile(item)) * 8192
                                          : p.ws[item_src(item)] + OFF_PK + (size_t)item_tile(item) * PK_TILE) + lane * 16;
            sample();
#pragma unroll
            for (int i = 0; i < 8; ++i) {
                if (!giveup) __builtin_amdgcn_global_load_lds((const unsigned*)(g + i * 1024), (LAS unsigned*)(lds + EX_MASK + k * 8192 + i * 1024), 16, 0, AUX_SYS);
                wait_vm<PULL_W>();
                if (i == PULL_W - 1 && k > 0) { if (lane == 0) *mland = (unsigned)k; }
            }
            absorb();
        }
        bool mpub = false;
        unsigned issued = 0, vpub = 0;
#pragma unroll 1
        for (int i = 0; i < 56; ++i) {
            const int k = i >> 3, wv = i & 7, item = (int)((order >> (4 * k)) & 7u);
            if (wv == 0 && !((known >> (8 + item)) & 1u)) {
                const unsigned long long t0 = rtime();
                unsigned it = 0;
                for (;;) {
                    poll();
                    vpub = issued;
                    if (lane == 0) { *vland = vpub; *mland = 7u; }
                    if (((known >> (8 + item)) & 1u) || giveup) break;
                    giveup = wait_giveup(myws, call, t0, it);
                }
                mpub = true;
            }
            if (wv == 2) sample();
            if (wv == 7) absorb();
            if (!mpub && lens[i] == 0u) { wait_vm<0>(); if (lane == 0) *mland = 7u; mpub = true; }
            lds_wait_ge(lens + i, 1u);
            const unsigned len = 112u * pack_rows(__builtin_amdgcn_readfirstlane(lens[i]) - 1u);
            const unsigned n = (len + 1023u) >> 10;
            if (lane == 0) endq[i] = issued + n + 1u;
            if (i >= 8 && done[i - 8] == 0u) {
                wait_vm<0>();
                vpub = issued;
                if (lane == 0) { *vland = vpub; *mland = 7u; }
                mpub = true;
                lds_wait_ge(done + (i - 8), 1u);
            }
            const unsigned char* g = p.ws[item_src(item)] + OFF_PK + (size_t)item_tile(item) * PK_TILE + 8192 + (size_t)wv * 8192 + lane * 16;
#pragma unroll 1
            for (unsigned it = 0; it < n; ++it) {
                if (!giveup && it * 1024u + lane * 16u < len)
                    __builtin_amdgcn_global_load_lds((const unsigned*)(g + it * 1024), (LAS unsigned*)(lds + EX_RING + (i & 7) * 8192 + it * 1024), 16, 0, AUX_SYS);
                ++issued;
                wait_vm<PULL_W>();
                if (issued >= (unsigned)PULL_W) {
                    if (issued - (unsigned)PULL_W > vpub) { vpub = issued - (unsigned)PULL_W; if (lane == 0) *vland = vpub; }
                    if (!mpub) { if (lane == 0) *mland = 7u; mpub = true; }
                }
            }
        }
        wait_vm<0>();
        if (lane == 0) { *mland = 7u; *vland = issued; }
        return;
    }
    {
        const int k = v - 1;
        lds_wait_ge(mland, (unsigned)(k + 1));
#pragma unroll
        for (int wv = 0; wv < 8; ++wv) {
            const u32x4 m = *(const LAS u32x4*)(lds + EX_MASK + k * 8192 + wv * 1024 + lane * 16);
            unsigned c = __builtin_popcount(m[0]) + __builtin_popcount(m[1]) + __builtin_popcount(m[2]) + __builtin_popcount(m[3]);
#pragma unroll
            for (int o = 32; o >= 1; o >>= 1) c += __shfl_xor(c, o);
            if (lane == 0) lens[k * 8 + wv] = c + 1u;
        }
    }
    const int fr = lane & 15, fq = lane >> 4;
    const __amdgpu_buffer_rsrc_t ors = mkrsrc(p.out[rank], (unsigned)OUT_BYTES);
#pragma unroll 1
    for (int i = v - 1; i < 56; i += 7) {
        const int k = i >> 3, wv = i & 7;
        lds_wait_ge(endq + i, 1u);
        lds_wait_ge(vland, __builtin_amdgcn_readfirstlane(endq[i]) - 1u);
        const unsigned tmo = ld_agent(myws + OFF_TMO);
        const int item = (int)(__builtin_amdgcn_readfirstlane(ordr[k]) & 7u);
        const int s = item_src(item), tile = item_tile(item), ti = tile >> 3, tj = tile & 7;
        u32x4 m = *(const LAS u32x4*)(lds + EX_MASK + k * 8192 + wv * 1024 + lane * 16);
        unsigned base = EX_RING + (i & 7) * 8192;
        {
            const unsigned R = pack_rows(__builtin_amdgcn_readfirstlane(lens[i]) - 1u);
            if ((unsigned)lane < R) {
                u32x4 r7 = {0u, 0u, 0u, 0u};
#pragma unroll
                for (int kk = 0; kk < 7; ++kk) {
                    const u32x4 d = *(const LAS u32x4*)(lds + base + (kk * R + lane) * 16);
#pragma unroll
                    for (int c = 0; c < 4; ++c) r7[c] |= ((d[c] >> 7) & 0x01010101u) << kk;
                }
                *(LAS u32x4*)(lds + base + (7 * R + lane) * 16) = r7;
            }
        }
        const unsigned obase = (unsigned)(ti * 256 + (wv >> 2) * 64 + fr) * (unsigned)NN + (unsigned)(s * NS + tj * 256 + (wv & 3) * 64 + fq * 16);
#pragma unroll
        for (int q = 0; q < 4; ++q) {
            const unsigned h = m[q];
            const unsigned pc = (unsigned)__builtin_popcount(h >> 16) | ((unsigned)__builtin_popcount(h & 0xffffu) << 16);
            const unsigned inc = wave_scan(pc), exc = inc - pc;
            const unsigned tot = (unsigned)__builtin_amdgcn_readlane((int)inc, 63);
#pragma unroll
            for (int hb = 0; hb < 2; ++hb) {
                const int blk = q * 2 + hb;
                unsigned addr = base + (hb == 0 ? (exc & 0xffffu) : (tot & 0xffffu) + (exc >> 16));
                u32x4 o;
#pragma unroll
                for (int jj = 0; jj < 4; ++jj) {
                    const unsigned m4 = (h >> (28 - 16 * hb - 4 * jj)) & 15u;
                    const unsigned sel = *(const LAS unsigned*)(lds + LDS_LUTR + (((addr & 3u) << 4) | m4) * 4);
                    const LAS unsigned* wp = (const LAS unsigned*)(lds + (addr & ~3u));
                    o[jj] = __builtin_amdgcn_perm(wp[1], wp[0], sel) & 0x7f7f7f7fu;
                    addr += (unsigned)__builtin_popcount(m4);
                }
                if (tmo == call) o = u32x4{0x7f7f7f7fu, 0x7f7f7f7fu, 0x7f7f7f7fu, 0x7f7f7f7fu};
                __builtin_amdgcn_raw_buffer_store_b128(o, ors, obase + (unsigned)((blk >> 2) * 128 + (blk & 3) * 16) * (unsigned)NN, 0, AUX_SYS);
            }
            base += (tot & 0xffffu) + (tot >> 16);
        }
        WAIT_L(0);
        if (lane == 0) done[i] = 1u;
    }
}

struct Hl0 { f32x4 lx[2], lw[8]; };
__device__ __forceinline__ void hl0_load(const Params& p, int dd, int j, int rg, int lane, Hl0& h) {
    const int q = (p.rank + dd) & 7;
    const float* xs = p.x + (size_t)(256 * j + 32 * dd + 16 * rg + (lane & 15)) * KS + 8 * (lane >> 4);
    const float* wsrc = p.w + (size_t)(8 * (lane >> 4)) * NN + q * NS + j * 128 + rg * 64 + 4 * (lane & 15);
#pragma unroll
    for (int i = 0; i < 8; ++i) h.lw[i] = __builtin_nontemporal_load((const f32x4*)(wsrc + (size_t)i * NN));
#pragma unroll
    for (int i = 0; i < 2; ++i) h.lx[i] = __builtin_nontemporal_load((const f32x4*)(xs + 4 * i));
}
__device__ __forceinline__ void hl0_push(const Params& p, int dd, int j, int rg, int lane, const Hl0& h) {
    const int rank = p.rank, q = (rank + dd) & 7;
    const unsigned kfs = rank & 1, tm = rank >> 1;
    const __amdgpu_buffer_rsrc_t wq = mkrsrc(p.ws[q] + OFF_WS, (unsigned)WS_BYTES);
    const unsigned wf = tm * (256u * 1024u) + (unsigned)(j >> 1) * 32768u + (unsigned)((j & 1) * 2 + rg) * 4096u + kfs * 1024u + lane * 16u;
    const unsigned xf = tm * (512u * 1024u) + (unsigned)(16 * j + 2 * dd + rg) * 2048u + kfs * 1024u + lane * 16u;
#pragma unroll
    for (int f = 0; f < 4; ++f) {
        const u32x4 cw = {pk_bf16(h.lw[0][f], h.lw[1][f]), pk_bf16(h.lw[2][f], h.lw[3][f]), pk_bf16(h.lw[4][f], h.lw[5][f]), pk_bf16(h.lw[6][f], h.lw[7][f])};
        __builtin_amdgcn_raw_buffer_store_b128(cw, wq, wf + (unsigned)((f >> 1) * 16384 + (f & 1) * 2048), 0, AUX_SYS);
        wait_vm<HL0_W>();
    }
    const u32x4 cx = {pk_bf16(h.lx[0][0], h.lx[0][1]), pk_bf16(h.lx[0][2], h.lx[0][3]), pk_bf16(h.lx[1][0], h.lx[1][1]), pk_bf16(h.lx[1][2], h.lx[1][3])};
#pragma unroll
    for (int k = 0; k < WORLD; ++k) {
        __builtin_amdgcn_raw_buffer_store_b128(cx, mkrsrc(p.ws[(rank + dd + k) & 7] + OFF_XS, (unsigned)XS_BYTES), xf, 0, AUX_SYS);
        wait_vm<HL0_W>();
    }
}
__device__ __forceinline__ void hl0_flags(const Params& p, LAS unsigned char* lds, int dd, int j, int lane, unsigned call) {
    const int rank = p.rank, q = (rank + dd) & 7;
    wait_vm<0>();
    unsigned old = 0;
    if (lane == 0) old = __hip_atomic_fetch_add((LAS unsigned*)(lds + LDS_MISC + 192), 1u, __ATOMIC_RELAXED, __HIP_MEMORY_SCOPE_WORKGROUP);
    if (__builtin_amdgcn_readfirstlane(old) == 1u) {
        unsigned g = 0;
        if (lane == 0) {
            st_sys(p.ws[q] + OFF_INFLAG + (size_t)(rank * 16) * 256 + (32 + j) * 4, call);
            g = __hip_atomic_fetch_add((gu32*)(p.ws[rank] + OFF_CNT0 + (j * 2 + (dd >> 2)) * 4), 1u, __ATOMIC_RELAXED, __HIP_MEMORY_SCOPE_AGENT);
        }
        if ((__builtin_amdgcn_readfirstlane(g) & 3u) == 3u) {
            if (lane < WORLD) st_sys(p.ws[lane] + OFF_INFLAG + (size_t)(rank * 16) * 256 + ((dd >> 2) * 16 + j) * 4, call);
        }
    }
}

__device__ __forceinline__ void gemm_role(const Params& p, LAS unsigned char* lds, int ti, int tj, unsigned call) {
    const int tid = threadIdx.x, wid = __builtin_amdgcn_readfirstlane(tid >> 6), lane = tid & 63;
    const int wr = wid >> 2, wc = wid & 3, fr = lane & 15, fq = lane >> 4;
    const int rank = p.rank;
    unsigned char* myws = p.ws[rank];
    const char* xsb = (const char*)myws + OFF_XS + (size_t)ti * 32768;
    const char* wsb = (const char*)myws + OFF_WS + (size_t)tj * 32768;
    const unsigned voff = (unsigned)tid * 16u;
    const unsigned ldsw = (unsigned)wid * 1024u;
    const int aoff = wr * 8192 + lane * 16, boff = wc * 4096 + lane * 16;
    const int fl8 = lane < 8 ? lane : 0, fw = fl8 & 3;
    const unsigned fl_off = (unsigned)(fl8 >> 2) * 4096u + (unsigned)(fw < 2 ? fw * 16 + ti : 32 + 2 * tj + (fw - 2)) * 4u;
    const unsigned char* flbase = myws + OFF_INFLAG + fl_off;
    auto ktile = [&](int t) { return (t & ~3) | (((ti >> 2) - t) & 3); };
    auto flag_addr = [&](int t) { const int T = ktile(t); return flbase + (size_t)(T & 3) * 8192 + (size_t)(T >> 2) * 256; };

#define SA(b, h) (((b) * 2 + (h)) * HTB)
#define SB(b, h) ((4 + (b) * 2 + (h)) * HTB)
#define STAGE(bufoff, gbase) do { _Pragma("unroll") for (int _i = 0; _i < 2; ++_i) \
        __builtin_amdgcn_global_load_lds((const unsigned*)((const char*)(gbase) + voff + _i * 8192), (LAS unsigned*)(lds + (bufoff) + ldsw + _i * 8192), 16, 0, AUX_SYS); } while (0)
#define LDA(dst, b, h) do { _Pragma("unroll") for (int m = 0; m < 4; ++m) _Pragma("unroll") for (int k = 0; k < 2; ++k) dst[m][k] = *(const LAS bf16x8*)(lds + SA(b, h) + aoff + m * 2048 + k * 1024); } while (0)
#define LDB(dst, b, h) do { _Pragma("unroll") for (int n = 0; n < 2; ++n) _Pragma("unroll") for (int k = 0; k < 2; ++k) dst[n][k] = *(const LAS bf16x8*)(lds + SB(b, h) + boff + n * 2048 + k * 1024); } while (0)
#define MMA(ai, bj, At, Bt) do { __builtin_amdgcn_s_setprio(1); _Pragma("unroll") for (int m = 0; m < 4; ++m) _Pragma("unroll") for (int n = 0; n < 2; ++n) _Pragma("unroll") for (int k = 0; k < 2; ++k) \
        acc[ai][bj][m][n] = __builtin_amdgcn_mfma_f32_16x16x32_bf16(Bt[n][k], At[m][k], acc[ai][bj][m][n], 0, 0, 0); __builtin_amdgcn_s_setprio(0); } while (0)
#define FLAG_SPIN(t) do { const unsigned char* _fp = flag_addr(t); const unsigned long long _t0 = rtime(); unsigned _it = 0; \
        for (;;) { const unsigned _v = ld_sys(_fp); if (__all((int)(_v - call) >= 0)) break; if (wait_giveup(myws, call, _t0, _it)) break; } } while (0)
#define FLAG_FETCH(t, s) __builtin_amdgcn_global_load_lds((const unsigned*)flag_addr(t), (LAS unsigned*)(lds + LDS_FLAG + (s) * 256), 4, 0, AUX_SYS)
#define FLAG_READ(s) (*(volatile LAS unsigned*)(lds + LDS_FLAG + (s) * 256 + lane * 4))
#define FLAG_CHECK(t, fv) do { if (!__all((int)((fv) - call) >= 0)) FLAG_SPIN(t); } while (0)

    f32x4 acc[2][2][4][2];
#pragma unroll
    for (int a = 0; a < 2; ++a)
#pragma unroll
        for (int b = 0; b < 2; ++b)
#pragma unroll
            for (int m = 0; m < 4; ++m)
#pragma unroll
                for (int n = 0; n < 2; ++n) acc[a][b][m][n] = (f32x4){0.f, 0.f, 0.f, 0.f};
    bf16x8 At[4][2], B0[2][2], B1[2][2];

    {
        if (wid == 0) { FLAG_SPIN(0); FLAG_SPIN(1); FLAG_FETCH(2, 0); }
        BAR;
        const char* a0 = xsb + (size_t)ktile(0) * (512 * KiB); const char* b0 = wsb + (size_t)ktile(0) * (256 * KiB);
        const char* a1 = xsb + (size_t)ktile(1) * (512 * KiB); const char* b1 = wsb + (size_t)ktile(1) * (256 * KiB);
        STAGE(SB(0, 0), b0); STAGE(SA(0, 0), a0); STAGE(SB(0, 1), b0 + HTB); STAGE(SA(0, 1), a0 + HTB);
        if (wr == 1) BAR;
        WAIT_V(4); BAR;
        STAGE(SB(1, 0), b1); STAGE(SA(1, 0), a1); STAGE(SB(1, 1), b1 + HTB);
        WAIT_V(6); BAR;
    }
    for (int t = 0; t < NTILES; t += 2) {
        const int t2 = t + 2 < NTILES ? t + 2 : t, t3 = t + 3 < NTILES ? t + 3 : t + 1, t4 = t + 4 < NTILES ? t + 4 : t;
        const char* a1 = xsb + (size_t)ktile(t + 1) * (512 * KiB);
        const char* a2 = xsb + (size_t)ktile(t2) * (512 * KiB); const char* b2 = wsb + (size_t)ktile(t2) * (256 * KiB);
        const char* a3 = xsb + (size_t)ktile(t3) * (512 * KiB); const char* b3 = wsb + (size_t)ktile(t3) * (256 * KiB);
        unsigned fv = FLAG_READ(0); SCHED;
        LDB(B0, 0, 0); SCHED; LDA(At, 0, 0); STAGE(SA(1, 1), a1 + HTB);
        if (wid == 0) { FLAG_CHECK(t2, fv); FLAG_FETCH(t3, 1); }
        WAIT_L(8); BAR; WAIT_L(0); MMA(0, 0, At, B0); BAR; SCHED;
        LDB(B1, 0, 1); STAGE(SB(0, 0), b2);
        BAR; WAIT_L(0); MMA(0, 1, At, B1); BAR;
        LDA(At, 0, 1); STAGE(SA(0, 0), a2);
        BAR; WAIT_L(0); MMA(1, 0, At, B0); BAR; SCHED;
        STAGE(SB(0, 1), b2 + HTB);
        WAIT_V(6); BAR; MMA(1, 1, At, B1); BAR;
        fv = FLAG_READ(1); SCHED;
        LDB(B0, 1, 0); SCHED; LDA(At, 1, 0); STAGE(SA(0, 1), a2 + HTB);
        if (wid == 0) { FLAG_CHECK(t3, fv); FLAG_FETCH(t4, 0); }
        WAIT_L(8); BAR; WAIT_L(0); MMA(0, 0, At, B0); BAR; SCHED;
        LDB(B1, 1, 1); STAGE(SB(1, 0), b3);
        BAR; WAIT_L(0); MMA(0, 1, At, B1); BAR;
        LDA(At, 1, 1); STAGE(SA(1, 0), a3);
        BAR; WAIT_L(0); MMA(1, 0, At, B0); BAR; SCHED;
        STAGE(SB(1, 1), b3 + HTB);
        WAIT_V(6); BAR; MMA(1, 1, At, B1); BAR;
    }
    WAIT_V(0);
    if (wr == 0) BAR;
    BAR;
    const int tb = ti * 8 + tj;
    unsigned char* pk = myws + OFF_PK + (size_t)tb * PK_TILE;

    u32x4 mk;
#pragma unroll
    for (int q = 0; q < 4; ++q) {
        unsigned m = 0;
#pragma unroll
        for (int n = 0; n < 32; ++n) {
            const int blk = q * 2 + (n >> 4), pos = n & 15, j = pos >> 2, f = pos & 3;
            m = __builtin_amdgcn_alignbit(m, __float_as_uint(acc[blk >> 2][f >> 1][blk & 3][f & 1][j]), 31);
        }
        mk[q] = ~m;
    }
    __builtin_amdgcn_raw_buffer_store_b128(mk, mkrsrc(pk, 8192), (unsigned)tid * 16u, 0, AUX_SYS);
    float mx = 0.f;
#pragma unroll
    for (int a = 0; a < 2; ++a)
#pragma unroll
        for (int b = 0; b < 2; ++b)
#pragma unroll
            for (int m = 0; m < 4; ++m)
#pragma unroll
                for (int n = 0; n < 2; ++n)
#pragma unroll
                    for (int j = 0; j < 4; ++j) mx = fmaxf(mx, acc[a][b][m][n][j]);
#pragma unroll
    for (int o = 32; o >= 1; o >>= 1) mx = fmaxf(mx, __shfl_xor(mx, o));
    LAS float* wmax = (LAS float*)(lds + LDS_MISC + 64);
    if (lane == 0) wmax[wid] = mx;
    WAIT_L(0); BAR;
    if (tid < WORLD) {
        float m8 = 0.f;
#pragma unroll
        for (int i = 0; i < 8; ++i) m8 = fmaxf(m8, wmax[i]);
        __hip_atomic_store((gu64*)(p.ws[tid] + OFF_AMAX + (size_t)(rank * 128 + tb) * 8),
                           ((unsigned long long)call << 32) | __float_as_uint(m8), __ATOMIC_RELAXED, __HIP_MEMORY_SCOPE_SYSTEM);
    }
#pragma unroll
    for (int i = 0; i < MASK_PUSH; ++i)
        __builtin_amdgcn_raw_buffer_store_b128(mk, mkrsrc(p.ws[(rank + 1 + (tb + i) % 7) & 7] + OFF_MK + (size_t)(rank * 128 + tb) * 8192, 8192),
                                               (unsigned)tid * 16u, 0, AUX_SYS);
    LAS unsigned char* cbuf = lds + wid * 8192;
#pragma unroll
    for (int i = 0; i < 8; ++i) *(LAS u32x4*)(cbuf + i * 1024 + lane * 16) = u32x4{0u, 0u, 0u, 0u};
    if (wid != 0) wait_vm<MASK_PUSH>(); else wait_vm<MASK_PUSH + 1>();
    BAR;
    {
        const bool pushed = tid < WORLD && tid != rank && mask_pushed(rank, tid, tb);
        if (tid < WORLD && tid != rank && !pushed) st_sys(p.ws[tid] + OFF_PKFLAG + (size_t)tb * 128 + rank * 4, call);
        if (MASK_PUSH > 0) {
            WAIT_V(0);
            unsigned old = 0;
            if (lane == 0) old = __hip_atomic_fetch_add((LAS unsigned*)(lds + LDS_MISC + 196), 1u, __ATOMIC_RELAXED, __HIP_MEMORY_SCOPE_WORKGROUP);
            if (__builtin_amdgcn_readfirstlane(old) == 7u) {
                if (lane < WORLD && lane != rank && mask_pushed(rank, lane, tb)) st_sys(p.ws[lane] + OFF_PKFLAG + (size_t)tb * 128 + rank * 4, call);
            }
        }
    }
    float gm;
    {
        const __amdgpu_buffer_rsrc_t ars = mkrsrc(myws + OFF_AMAX, 8192);
        const unsigned long long t0 = rtime();
        u32x4 g;
        unsigned it = 0;
        for (;;) {
            g = __builtin_amdgcn_raw_buffer_load_b128(ars, tid * 16, 0, AUX_SYS);
            if (__all(g[1] == call && g[3] == call)) break;
            if (wait_giveup(myws, call, t0, it)) break;
        }
        gm = fmaxf(__uint_as_float(g[0]), __uint_as_float(g[2]));
#pragma unroll
        for (int o = 32; o >= 1; o >>= 1) gm = fmaxf(gm, __shfl_xor(gm, o));
    }
    LAS float* wmax2 = (LAS float*)(lds + LDS_MISC + 128);
    if (lane == 0) wmax2[wid] = gm;
    const unsigned tmo_w = ld_agent(myws + OFF_TMO);
    WAIT_L(0); BAR;
    float amax = 0.f;
#pragma unroll
    for (int i = 0; i < 8; ++i) amax = fmaxf(amax, wmax2[i]);
    const bool bad = tmo_w == call;
    const float scale = amax / 448.0f;
    const float inv = scale > 0.f ? 1.0f / scale : 0.f;

    u32x4 q8[8];
    unsigned cnt = 0;
#pragma unroll
    for (int q = 0; q < 4; ++q) {
        const unsigned h = mk[q];
        const unsigned pc = (unsigned)__builtin_popcount(h >> 16) | ((unsigned)__builtin_popcount(h & 0xffffu) << 16);
        const unsigned inc = wave_scan(pc), exc = inc - pc;
        const unsigned tot = (unsigned)__builtin_amdgcn_readlane((int)inc, 63);
#pragma unroll
        for (int hb = 0; hb < 2; ++hb) {
            const int blk = q * 2 + hb, a = blk >> 2, m = blk & 3;
            unsigned addr = (unsigned)(wid * 8192) + cnt + (hb == 0 ? (exc & 0xffffu) : (tot & 0xffffu) + (exc >> 16));
#pragma unroll
            for (int j = 0; j < 4; ++j) {
                const float v0 = __builtin_amdgcn_fmed3f(acc[a][0][m][0][j] * inv, 0.f, 448.f), v1 = __builtin_amdgcn_fmed3f(acc[a][0][m][1][j] * inv, 0.f, 448.f);
                const float v2 = __builtin_amdgcn_fmed3f(acc[a][1][m][0][j] * inv, 0.f, 448.f), v3 = __builtin_amdgcn_fmed3f(acc[a][1][m][1][j] * inv, 0.f, 448.f);
                int r = __builtin_amdgcn_cvt_pk_fp8_f32(v0, v1, 0, false);
                r = __builtin_amdgcn_cvt_pk_fp8_f32(v2, v3, r, true);
                const unsigned w = bad ? 0x7f7f7f7fu : (unsigned)r;
                q8[blk][j] = w;
                const unsigned m4 = (h >> (28 - 16 * hb - 4 * j)) & 15u;
                const u32x2 sel = *(const LAS u32x2*)(lds + LDS_LUTS + (((addr & 3u) << 4) | m4) * 8);
                LAS unsigned* wp = (LAS unsigned*)(lds + (addr & ~3u));
                __hip_atomic_fetch_or(wp, __builtin_amdgcn_perm(0u, w, sel[0]), __ATOMIC_RELAXED, __HIP_MEMORY_SCOPE_WORKGROUP);
                __hip_atomic_fetch_or(wp + 1, __builtin_amdgcn_perm(0u, w, sel[1]), __ATOMIC_RELAXED, __HIP_MEMORY_SCOPE_WORKGROUP);
                addr += (unsigned)__builtin_popcount(m4);
            }
        }
        cnt += (tot & 0xffffu) + (tot >> 16);
    }
    {
        const unsigned R = pack_rows(cnt);
        const __amdgpu_buffer_rsrc_t vrs = mkrsrc(pk + 8192 + wid * 8192, 8192);
        if ((unsigned)lane < R) {
            const u32x4 r7 = *(const LAS u32x4*)(cbuf + (7 * R + lane) * 16);
#pragma unroll
            for (int kk = 0; kk < 7; ++kk) {
                u32x4 d = *(const LAS u32x4*)(cbuf + (kk * R + lane) * 16);
#pragma unroll
                for (int c = 0; c < 4; ++c) d[c] = (d[c] & 0x7f7f7f7fu) | (((r7[c] >> kk) & 0x01010101u) << 7);
                __builtin_amdgcn_raw_buffer_store_b128(d, vrs, (kk * R + lane) * 16, 0, AUX_SYS);
            }
        }
        WAIT_V(0);
        BAR;
        if (tid < WORLD && tid != rank) st_sys(p.ws[tid] + OFF_PKFLAG + (size_t)tb * 128 + 32 + rank * 4, call);
    }
    {
        const unsigned obase = (unsigned)(ti * 256 + wr * 64 + fr) * (unsigned)NN + (unsigned)(rank * NS + tj * 256 + wc * 64 + fq * 16);
        const __amdgpu_buffer_rsrc_t ors = mkrsrc(p.out[rank], (unsigned)OUT_BYTES);
#pragma unroll
        for (int blk = 0; blk < 8; ++blk)
            __builtin_amdgcn_raw_buffer_store_b128(q8[blk], ors, obase + (unsigned)((blk >> 2) * 128 + (blk & 3) * 16) * (unsigned)NN, 0, AUX_SYS);
    }
#undef SA
#undef SB
#undef STAGE
#undef LDA
#undef LDB
#undef MMA
}

__global__ __launch_bounds__(512, 2) void fused_gemm_dist(Params p) {
    extern __shared__ __attribute__((aligned(16))) unsigned char shm[];
    LAS unsigned char* lds = (LAS unsigned char*)shm;
    const int tid = threadIdx.x;
    const int xcd = blockIdx.x & 7, idx = blockIdx.x >> 3;
    unsigned char* myws = p.ws[p.rank];
    if (tid < 352) ((LAS unsigned*)(lds + LDS_MISC))[tid] = 0u;
    if (tid == 352 && idx < 16) *(LAS unsigned*)(lds + LDS_CALL) = 0u;
    if (tid >= 448) init_luts(lds, tid - 448);
    unsigned ticket = 0;
    const int tk = idx >= 16 ? 128 : 0;
    if (tid == tk) ticket = __hip_atomic_fetch_add((gu32*)(myws + OFF_TICKET), 1u, __ATOMIC_RELAXED, __HIP_MEMORY_SCOPE_AGENT);
    volatile LAS unsigned* callp = (volatile LAS unsigned*)(lds + LDS_CALL);
    if (idx >= 16) {
        const int r = idx - 16;
        const int wv = __builtin_amdgcn_readfirstlane(tid >> 6);
        Hl0 h0;
        if (wv < 2) { hl0_load(p, r >> 1, xcd + 8 * (r & 1), wv, tid & 63, h0); hl0_push(p, r >> 1, xcd + 8 * (r & 1), wv, tid & 63, h0); }
        if (tid == tk) *callp = ticket / NWG + 1u;
        __syncthreads();
        const unsigned call = *callp;
        if (wv < 2) hl0_flags(p, lds, r >> 1, xcd + 8 * (r & 1), tid & 63, call);
        gemm_role(p, lds, (xcd >> 1) * 4 + (r & 3), (xcd & 1) * 4 + (r >> 2), call);
    } else {
        WAIT_L(0);
        comm_inputs(p, lds, idx >> 1, xcd + 8 * (idx & 1), idx * 8 + xcd, ticket);
        __syncthreads();
        const unsigned call = *callp;
        comm_expand(p, lds, idx * 8 + xcd, call);
        __syncthreads();
    }
}

}

extern "C" void kernel_launch(int rank, int nranks, ncclComm_t comm, hipStream_t stream,
                              void* const* d_in, const int* in_sizes, int n_in,
                              void* d_out, int out_size, const int* in_dtypes, int out_dtype,
                              void* const* d_out_all, void* const* d_ws_all, size_t ws_bytes) {
    (void)comm; (void)in_dtypes; (void)out_dtype; (void)d_out;
    if (nranks != WORLD || n_in < 2 || (unsigned)rank >= (unsigned)WORLD || ws_bytes < WS_NEEDED ||
        in_sizes[0] != MM * KS || in_sizes[1] != KS * NN || (size_t)out_size != OUT_BYTES) {
        fprintf(stderr, "kernel_launch: unexpected problem shape\n");
        return;
    }
    static bool attr_done[64] = {};
    int dev = 0;
    HIP_CHECK(hipGetDevice(&dev));
    if (dev >= 0 && dev < 64 && !attr_done[dev]) {
        HIP_CHECK(hipFuncSetAttribute((const void*)fused_gemm_dist, hipFuncAttributeMaxDynamicSharedMemorySize, LDS_BYTES));
        attr_done[dev] = true;
    }
    Params p;
    p.x = (const float*)d_in[0];
    p.w = (const float*)d_in[1];
    for (int i = 0; i < WORLD; ++i) { p.out[i] = (unsigned char*)d_out_all[i]; p.ws[i] = (unsigned char*)d_ws_all[i]; }
    p.rank = rank;
    hipLaunchKernelGGL(fused_gemm_dist, dim3(NWG), dim3(512), LDS_BYTES, stream, p);
    HIP_CHECK(hipGetLastError());
}
```
